```python
import numpy as np

kernel_name = "gpu_a2a_gemm_m2048_k2048_n2048_bf16_silu"


ABSMAX_RTOL = 0.008

COMPUTE_DTYPE = "bfloat16"
EPILOGUE = "silu"
IN_QUANT = None
OUT_QUANT = None
IN_DTYPES = {"x": "bfloat16", "w_mat": "bfloat16"}
WORLD_SIZE = 8

def _to_bf16(x):
    u = np.ascontiguousarray(x, dtype=np.float32).view(np.uint32)
    return ((u + 0x7FFF + ((u >> 16) & 1)) & 0xFFFF0000).view(np.float32)

def setup_inputs():

    x = _to_bf16(np.random.randn(2048, 2048).astype(np.float32))
    w_mat = _to_bf16((np.random.randn(2048, 2048) / np.sqrt(2048)).astype(np.float32))
    return {"x": x, "w_mat": w_mat}

def _epilogue(y):
    return (y / (1.0 + np.exp(-np.clip(y, -60.0, 60.0)))).astype(np.float32)

def reference(x, w_mat):
    y = (x @ w_mat).astype(np.float32)
    return _epilogue(y)
```

```cpp
#include <hip/hip_runtime.h>
#include <rccl/rccl.h>
#include <cstdio>
#include <cstdint>

typedef short bf16x8 __attribute__((ext_vector_type(8)));
typedef short s16x4 __attribute__((ext_vector_type(4)));
typedef float f32x16 __attribute__((ext_vector_type(16)));
typedef float f32x4 __attribute__((ext_vector_type(4)));
typedef unsigned u32x4 __attribute__((ext_vector_type(4)));

constexpr int NR = 8;
constexpr int KB = 256;
constexpr int NN = 2048;
constexpr size_t SLOT = 256 * 256 * 2;
constexpr size_t PAR = NR * SLOT;
constexpr size_t FLAG_OFF = 2 * PAR;
constexpr int NPIECE = 16;
constexpr size_t CTR_OFF = FLAG_OFF + (size_t)NR * NPIECE * 128;
constexpr size_t TMO_OFF = CTR_OFF + 2048 * 4;
constexpr unsigned long long DEADLINE_TICKS = 50000000ull;

__device__ __forceinline__ unsigned short f2bf(float f) {
  unsigned u = __float_as_uint(f);
  return (unsigned short)((u + 0x7FFFu + ((u >> 16) & 1u)) >> 16);
}

__global__ __launch_bounds__(512, 2) void a2a_gemm_kernel(int rank, const short* __restrict__ x,
                                                          const short* __restrict__ w,
                                                          short* __restrict__ out,
                                                          void* const* __restrict__ ws_all) {
  __shared__ __attribute__((aligned(16))) char smem[128 * 1024];
  __shared__ int s_bad;
  const int b = blockIdx.x;
  const int tid = threadIdx.x;
  const int wv = __builtin_amdgcn_readfirstlane(tid >> 6);
  const int lane = tid & 63;
  char* myws = (char*)ws_all[rank];
  if (tid == 0) s_bad = 0;

  unsigned call;
  {
    unsigned* ctr = (unsigned*)(myws + CTR_OFF) + (b * 8 + wv);
    unsigned old = 0;
    if (lane == 0) old = __hip_atomic_fetch_add(ctr, 1u, __ATOMIC_RELAXED, __HIP_MEMORY_SCOPE_AGENT);
    call = __builtin_amdgcn_readfirstlane(old) + 1u;
  }
  const size_t par = (size_t)(call & 1u) * PAR;

  if (wv == rank && b < 7 * NPIECE) {
    const int d = (rank + 1 + (b % 7)) & 7;
    const int piece = b / 7;
    const u32x4* s = (const u32x4*)(x + (size_t)(d * 256 + piece * 16) * KB);
    char* dws = (char*)ws_all[d];
    u32x4* t = (u32x4*)(dws + par + (size_t)rank * SLOT + (size_t)piece * 8192);
    u32x4 v[8];
#pragma unroll
    for (int i = 0; i < 8; ++i) v[i] = s[i * 64 + lane];
#pragma unroll
    for (int i = 0; i < 8; ++i) t[i * 64 + lane] = v[i];
    asm volatile("s_waitcnt vmcnt(0)" ::: "memory");
    if (lane == 0) {
      __builtin_amdgcn_fence(__ATOMIC_RELEASE, "");
      asm volatile("s_waitcnt vmcnt(0)" ::: "memory");
      unsigned* flag = (unsigned*)(dws + FLAG_OFF + (size_t)(rank * NPIECE + piece) * 128);
      __hip_atomic_store(flag, call, __ATOMIC_RELAXED, __HIP_MEMORY_SCOPE_SYSTEM);
    }
  }

  const int xcd = b & 7, idx = b >> 3;
  const int cg = xcd * 4 + (idx & 3), rg = idx >> 2;
  const int n0 = cg * 64;
  const int r = lane & 31, h = lane >> 5;

  char* sw = smem + wv * 16384;
  bf16x8 Bf[4][4][2];
#pragma unroll
  for (int half = 0; half < 2; ++half) {
    asm volatile("s_waitcnt lgkmcnt(0)" ::: "memory");
#pragma unroll
    for (int i = 0; i < 16; ++i) {
      const int row = i * 8 + (lane >> 3), col = (lane & 7) * 8;
      __builtin_amdgcn_global_load_lds(
          (const __attribute__((address_space(1))) void*)(w + (size_t)(KB * wv + half * 128 + row) * NN + n0 + col),
          (__attribute__((address_space(3))) void*)(sw + i * 1024), 16, 0, 0);
    }
    asm volatile("s_waitcnt vmcnt(0)" ::: "memory");
    const int g = lane >> 4, i16 = lane & 15, q = i16 >> 2, p = i16 & 3;
#pragma unroll
    for (int c2 = 0; c2 < 2; ++c2)
#pragma unroll
      for (int s = 0; s < 4; ++s)
#pragma unroll
        for (int nt = 0; nt < 2; ++nt) {
          const int k0 = c2 * 64 + 32 * (g >> 1) + 8 * s;
          char* a1 = sw + (k0 + q) * 128 + (nt * 32 + (g & 1) * 16 + 4 * p) * 2;
          s16x4 lo = __builtin_amdgcn_ds_read_tr16_b64_v4i16((__attribute__((address_space(3))) s16x4*)(a1));
          s16x4 hi = __builtin_amdgcn_ds_read_tr16_b64_v4i16((__attribute__((address_space(3))) s16x4*)(a1 + 512));
          bf16x8 f;
          f[0] = lo[0]; f[1] = lo[1]; f[2] = lo[2]; f[3] = lo[3];
          f[4] = hi[0]; f[5] = hi[1]; f[6] = hi[2]; f[7] = hi[3];
          Bf[half * 2 + c2][s][nt] = f;
        }
  }

  const short* abase;
  if (wv == rank) {
    abase = x + (size_t)(rank * 256) * KB;
  } else {
    abase = (const short*)(myws + par + (size_t)wv * SLOT);
    unsigned* tmo = (unsigned*)(myws + TMO_OFF);
    if (lane == 0) {
      const unsigned long long t0 = __builtin_amdgcn_s_memrealtime();
      for (int pc = 0; pc < 2; ++pc) {
        unsigned* flag = (unsigned*)(myws + FLAG_OFF + (size_t)(wv * NPIECE + rg * 2 + pc) * 128);
        while (__hip_atomic_load(flag, __ATOMIC_RELAXED, __HIP_MEMORY_SCOPE_SYSTEM) < call) {
          if (__hip_atomic_load(tmo, __ATOMIC_RELAXED, __HIP_MEMORY_SCOPE_AGENT) == call ||
              __builtin_amdgcn_s_memrealtime() - t0 > DEADLINE_TICKS) {
            __hip_atomic_store(tmo, call, __ATOMIC_RELAXED, __HIP_MEMORY_SCOPE_AGENT);
            s_bad = 1;
            break;
          }
          __builtin_amdgcn_s_sleep(1);
        }
      }
    }
    __builtin_amdgcn_fence(__ATOMIC_ACQUIRE, "");
  }

  const short* ap = abase + (size_t)(rg * 32 + r) * KB + 32 * h;
  bf16x8 Af[4][4];
#pragma unroll
  for (int c = 0; c < 4; ++c)
#pragma unroll
    for (int s = 0; s < 4; ++s) Af[c][s] = *(const bf16x8*)(ap + 64 * c + 8 * s);

  f32x16 acc0 = {}, acc1 = {};
#pragma unroll
  for (int c = 0; c < 4; ++c)
#pragma unroll
    for (int s = 0; s < 4; ++s) {
      acc0 = __builtin_amdgcn_mfma_f32_32x32x16_bf16(Af[c][s], Bf[c][s][0], acc0, 0, 0, 0);
      acc1 = __builtin_amdgcn_mfma_f32_32x32x16_bf16(Af[c][s], Bf[c][s][1], acc1, 0, 0, 0);
    }

  __syncthreads();
  float* part = (float*)smem;
#pragma unroll
  for (int i = 0; i < 16; ++i) {
    const int row = (i & 3) + 8 * (i >> 2) + 4 * h;
    part[(wv * 32 + row) * 64 + r] = acc0[i];
    part[(wv * 32 + row) * 64 + 32 + r] = acc1[i];
  }
  __syncthreads();
  {
    const int row = tid >> 4, c4 = (tid & 15) * 4;
    f32x4 sum = *(const f32x4*)(part + row * 64 + c4);
#pragma unroll
    for (int k = 1; k < 8; ++k) sum += *(const f32x4*)(part + (k * 32 + row) * 64 + c4);
    const bool bad = s_bad != 0;
    s16x4 o;
#pragma unroll
    for (int e = 0; e < 4; ++e) {
      float y = sum[e];
      float z = fminf(fmaxf(y, -60.f), 60.f);
      float v = y / (1.f + __expf(-z));
      o[e] = bad ? (short)0x7FC0 : (short)f2bf(v);
    }
    *(s16x4*)(out + (size_t)(rg * 32 + row) * NN + n0 + c4) = o;
  }
}

extern "C" void kernel_launch(int rank, int nranks, ncclComm_t comm, hipStream_t stream,
                              void* const* d_in, const int* in_sizes, int n_in, void* d_out,
                              int out_size, const int* in_dtypes, int out_dtype,
                              void* const* d_out_all, void* const* d_ws_all, size_t ws_bytes) {
  (void)comm; (void)in_sizes; (void)n_in; (void)out_size; (void)in_dtypes; (void)out_dtype;
  (void)d_out_all; (void)ws_bytes; (void)nranks;
  hipLaunchKernelGGL(a2a_gemm_kernel, dim3(256), dim3(512), 0, stream, rank,
                     (const short*)d_in[0], (const short*)d_in[1], (short*)d_out, d_ws_all);
}
```

```cpp
#include <hip/hip_runtime.h>
#include <rccl/rccl.h>
#include <cstdio>
#include <cstdint>

typedef short bf16x8 __attribute__((ext_vector_type(8)));
typedef short s16x4 __attribute__((ext_vector_type(4)));
typedef float f32x16 __attribute__((ext_vector_type(16)));
typedef float f32x4 __attribute__((ext_vector_type(4)));
typedef unsigned u32x4 __attribute__((ext_vector_type(4)));

constexpr int NR = 8;
constexpr int KB = 256;
constexpr int NN = 2048;
constexpr size_t SLOT = 256 * 256 * 2;
constexpr size_t PAR = NR * SLOT;
constexpr size_t FLAG_OFF = 2 * PAR;
constexpr int NPIECE = 16;
constexpr size_t CTR_OFF = FLAG_OFF + (size_t)NR * NPIECE * 128;
constexpr size_t TMO_OFF = CTR_OFF + 2048 * 4;
#define STAMP(j)
constexpr unsigned long long DEADLINE_TICKS = 50000000ull;

__device__ __forceinline__ unsigned short f2bf(float f) {
  unsigned u = __float_as_uint(f);
  return (unsigned short)((u + 0x7FFFu + ((u >> 16) & 1u)) >> 16);
}

__global__ __launch_bounds__(512, 2) void a2a_gemm_kernel(int rank, const short* __restrict__ x,
                                                          const short* __restrict__ w,
                                                          short* __restrict__ out,
                                                          void* const* __restrict__ ws_all) {
  __shared__ __attribute__((aligned(16))) char smem[128 * 1024];
  __shared__ int s_bad;
  const int b = blockIdx.x;
  const int tid = threadIdx.x;
  const int wv = __builtin_amdgcn_readfirstlane(tid >> 6);
  const int lane = tid & 63;
  char* myws = (char*)ws_all[rank];
  if (tid == 0) s_bad = 0;
  STAMP(0);

  const bool pusher = b < 7 * NPIECE;
  const int pd = (rank + 1 + (b % 7)) & 7;
  const int piece = b / 7;
  u32x4 pv;
  if (pusher) pv = ((const u32x4*)(x + (size_t)(pd * 256 + piece * 16) * KB))[tid];

  unsigned call;
  {
    unsigned* ctr = (unsigned*)(myws + CTR_OFF) + (b * 8 + wv);
    unsigned old = 0;
    if (lane == 0) old = __hip_atomic_fetch_add(ctr, 1u, __ATOMIC_RELAXED, __HIP_MEMORY_SCOPE_AGENT);
    call = __builtin_amdgcn_readfirstlane(old) + 1u;
  }
  const size_t par = (size_t)(call & 1u) * PAR;
  if (pusher) {
    char* dws = (char*)ws_all[pd];
    u32x4* t = (u32x4*)(dws + par + (size_t)rank * SLOT + (size_t)piece * 8192) + tid;
    asm volatile("global_store_dwordx4 %0, %1, off sc0 sc1\n\ts_nop 1" ::"v"(t), "v"(pv) : "memory");
    asm volatile("s_waitcnt vmcnt(0)" ::: "memory");
    __syncthreads();
    if (tid == 0) {
      unsigned* flag = (unsigned*)(dws + FLAG_OFF + (size_t)(rank * NPIECE + piece) * 128);
      __hip_atomic_store(flag, call, __ATOMIC_RELAXED, __HIP_MEMORY_SCOPE_SYSTEM);
    }
  }

  STAMP(1);
  const int xcd = b & 7, idx = b >> 3;
#define CX 8
  const int cg = (xcd * CX + (idx % CX)) & 31, rg = ((xcd * CX) / 32) * (32 / CX) + idx / CX;
  const int n0 = cg * 64;
  const int r = lane & 31, h = lane >> 5;

  char* sw = smem + wv * 16384;
  bf16x8 Bf[4][4][2];
#pragma unroll
  for (int half = 0; half < 2; ++half) {
    asm volatile("s_waitcnt lgkmcnt(0)" ::: "memory");
#pragma unroll
    for (int i = 0; i < 16; ++i) {
      const int row = i * 8 + (lane >> 3), col = (lane & 7) * 8;
      __builtin_amdgcn_global_load_lds(
          (const __attribute__((address_space(1))) void*)(w + (size_t)(KB * wv + half * 128 + row) * NN + n0 + col),
          (__attribute__((address_space(3))) void*)(sw + i * 1024), 16, 0, 0);
    }
    asm volatile("s_waitcnt vmcnt(0)" ::: "memory");
    const int g = lane >> 4, i16 = lane & 15, q = i16 >> 2, p = i16 & 3;
#pragma unroll
    for (int c2 = 0; c2 < 2; ++c2)
#pragma unroll
      for (int s = 0; s < 4; ++s)
#pragma unroll
        for (int nt = 0; nt < 2; ++nt) {
          const int k0 = c2 * 64 + 32 * (g >> 1) + 8 * s;
          char* a1 = sw + (k0 + q) * 128 + (nt * 32 + (g & 1) * 16 + 4 * p) * 2;
          s16x4 lo = __builtin_amdgcn_ds_read_tr16_b64_v4i16((__attribute__((address_space(3))) s16x4*)(a1));
          s16x4 hi = __builtin_amdgcn_ds_read_tr16_b64_v4i16((__attribute__((address_space(3))) s16x4*)(a1 + 512));
          bf16x8 f;
          f[0] = lo[0]; f[1] = lo[1]; f[2] = lo[2]; f[3] = lo[3];
          f[4] = hi[0]; f[5] = hi[1]; f[6] = hi[2]; f[7] = hi[3];
          Bf[half * 2 + c2][s][nt] = f;
        }
  }

  STAMP(2);
  const short* abase;
  if (wv == rank) {
    abase = x + (size_t)(rank * 256) * KB;
  } else {
    abase = (const short*)(myws + par + (size_t)wv * SLOT);
    unsigned* tmo = (unsigned*)(myws + TMO_OFF);
    if (lane == 0) {
      const unsigned long long t0 = __builtin_amdgcn_s_memrealtime();
      for (int pc = 0; pc < 2; ++pc) {
        unsigned* flag = (unsigned*)(myws + FLAG_OFF + (size_t)(wv * NPIECE + rg * 2 + pc) * 128);
        while (__hip_atomic_load(flag, __ATOMIC_RELAXED, __HIP_MEMORY_SCOPE_SYSTEM) < call) {
          if (__hip_atomic_load(tmo, __ATOMIC_RELAXED, __HIP_MEMORY_SCOPE_AGENT) == call ||
              __builtin_amdgcn_s_memrealtime() - t0 > DEADLINE_TICKS) {
            __hip_atomic_store(tmo, call, __ATOMIC_RELAXED, __HIP_MEMORY_SCOPE_AGENT);
            s_bad = 1;
            break;
          }
          __builtin_amdgcn_s_sleep(1);
        }
      }
    }
#define AMODE 2
    __builtin_amdgcn_fence(__ATOMIC_ACQUIRE, "wavefront");
    asm volatile("" ::: "memory");
  }

  STAMP(3);
  const short* ap = abase + (size_t)(rg * 32 + r) * KB + 32 * h;
  bf16x8 Af[4][4];
#pragma unroll
  for (int c = 0; c < 4; ++c)
#pragma unroll
    for (int s = 0; s < 4; ++s) {
      Af[c][s] = *(const bf16x8*)(ap + 64 * c + 8 * s);
    }

  f32x16 acc0 = {}, acc1 = {};
#pragma unroll
  for (int c = 0; c < 4; ++c)
#pragma unroll
    for (int s = 0; s < 4; ++s) {
      acc0 = __builtin_amdgcn_mfma_f32_32x32x16_bf16(Af[c][s], Bf[c][s][0], acc0, 0, 0, 0);
      acc1 = __builtin_amdgcn_mfma_f32_32x32x16_bf16(Af[c][s], Bf[c][s][1], acc1, 0, 0, 0);
    }

  STAMP(4);
  __syncthreads();
  float* part = (float*)smem;
#pragma unroll
  for (int i = 0; i < 16; ++i) {
    const int row = (i & 3) + 8 * (i >> 2) + 4 * h;
    part[(wv * 32 + row) * 64 + r] = acc0[i];
    part[(wv * 32 + row) * 64 + 32 + r] = acc1[i];
  }
  __syncthreads();
  {
    const int row = tid >> 4, c4 = (tid & 15) * 4;
    f32x4 sum = *(const f32x4*)(part + row * 64 + c4);
#pragma unroll
    for (int k = 1; k < 8; ++k) sum += *(const f32x4*)(part + (k * 32 + row) * 64 + c4);
    const bool bad = s_bad != 0;
    s16x4 o;
#pragma unroll
    for (int e = 0; e < 4; ++e) {
      float y = sum[e];
      float z = fminf(fmaxf(y, -60.f), 60.f);
      float v = y / (1.f + __expf(-z));
      o[e] = bad ? (short)0x7FC0 : (short)f2bf(v);
    }
    *(s16x4*)(out + (size_t)(rg * 32 + row) * NN + n0 + c4) = o;
  }
  STAMP(5);
}

extern "C" void kernel_launch(int rank, int nranks, ncclComm_t comm, hipStream_t stream,
                              void* const* d_in, const int* in_sizes, int n_in, void* d_out,
                              int out_size, const int* in_dtypes, int out_dtype,
                              void* const* d_out_all, void* const* d_ws_all, size_t ws_bytes) {
  (void)comm; (void)in_sizes; (void)n_in; (void)out_size; (void)in_dtypes; (void)out_dtype;
  (void)d_out_all; (void)ws_bytes; (void)nranks;
  hipLaunchKernelGGL(a2a_gemm_kernel, dim3(256), dim3(512), 0, stream, rank,
                     (const short*)d_in[0], (const short*)d_in[1], (short*)d_out, d_ws_all);
}
```

```cpp
#include <hip/hip_runtime.h>
#include <rccl/rccl.h>
#include <cstdio>
#include <cstdint>

typedef short bf16x8 __attribute__((ext_vector_type(8)));
typedef short s16x4 __attribute__((ext_vector_type(4)));
typedef float f32x16 __attribute__((ext_vector_type(16)));
typedef float f32x4 __attribute__((ext_vector_type(4)));
typedef unsigned u32x4 __attribute__((ext_vector_type(4)));

constexpr int NR = 8;
constexpr int KB = 256;
constexpr int NN = 2048;
constexpr size_t SLOT = 256 * 256 * 2;
constexpr size_t PAR = NR * SLOT;
constexpr size_t FLAG_OFF = 2 * PAR;
constexpr int NPIECE = 16;
constexpr size_t CTR_OFF = FLAG_OFF + (size_t)NR * NPIECE * 128;
constexpr size_t TMO_OFF = CTR_OFF + 2048 * 4;
#define STAMP(j)
constexpr unsigned long long DEADLINE_TICKS = 50000000ull;

__device__ __forceinline__ unsigned short f2bf(float f) {
  unsigned u = __float_as_uint(f);
  return (unsigned short)((u + 0x7FFFu + ((u >> 16) & 1u)) >> 16);
}

__global__ __launch_bounds__(512, 2) void a2a_gemm_kernel(int rank, const short* __restrict__ x,
                                                          const short* __restrict__ w,
                                                          short* __restrict__ out,
                                                          void* const* __restrict__ ws_all) {
  __shared__ __attribute__((aligned(16))) char smem[128 * 1024];
  __shared__ int s_bad;
  const int b = blockIdx.x;
  const int tid = threadIdx.x;
  const int wv = __builtin_amdgcn_readfirstlane(tid >> 6);
  const int lane = tid & 63;
  char* myws = (char*)ws_all[rank];
  if (tid == 0) s_bad = 0;
  STAMP(0);

  const bool pusher = b < 7 * NPIECE;
  const int pd = (rank + 1 + (b % 7)) & 7;
  const int piece = b / 7;
  u32x4 pv;
  if (pusher) pv = ((const u32x4*)(x + (size_t)(pd * 256 + piece * 16) * KB))[tid];

  unsigned call;
  {
    unsigned* ctr = (unsigned*)(myws + CTR_OFF) + (b * 8 + wv);
    unsigned old = 0;
    if (lane == 0) old = __hip_atomic_fetch_add(ctr, 1u, __ATOMIC_RELAXED, __HIP_MEMORY_SCOPE_AGENT);
    call = __builtin_amdgcn_readfirstlane(old) + 1u;
  }
  const size_t par = (size_t)(call & 1u) * PAR;
  if (pusher) {
    char* dws = (char*)ws_all[pd];
    u32x4* t = (u32x4*)(dws + par + (size_t)rank * SLOT + (size_t)piece * 8192) + tid;
    asm volatile("global_store_dwordx4 %0, %1, off sc0 sc1\n\ts_nop 1" ::"v"(t), "v"(pv) : "memory");
    asm volatile("s_waitcnt vmcnt(0)" ::: "memory");
    __syncthreads();
    if (tid == 0) {
      unsigned* flag = (unsigned*)(dws + FLAG_OFF + (size_t)(rank * NPIECE + piece) * 128);
      __hip_atomic_store(flag, call, __ATOMIC_RELAXED, __HIP_MEMORY_SCOPE_SYSTEM);
    }
  }

  STAMP(1);
  const int xcd = b & 7, idx = b >> 3;
#define CX 8
  const int cg = (xcd * CX + (idx % CX)) & 31, rg = ((xcd * CX) / 32) * (32 / CX) + idx / CX;
  const int n0 = cg * 64;
  const int r = lane & 31, h = lane >> 5;

  char* sw = smem + wv * 16384;
  bf16x8 Bf[4][4][2];
#pragma unroll
  for (int half = 0; half < 2; ++half) {
    asm volatile("s_waitcnt lgkmcnt(0)" ::: "memory");
#pragma unroll
    for (int i = 0; i < 16; ++i) {
      const int row = i * 8 + (lane >> 3), col = (lane & 7) * 8;
      __builtin_amdgcn_global_load_lds(
          (const __attribute__((address_space(1))) void*)(w + (size_t)(KB * wv + half * 128 + row) * NN + n0 + col),
          (__attribute__((address_space(3))) void*)(sw + i * 1024), 16, 0, 0);
    }
    asm volatile("s_waitcnt vmcnt(0)" ::: "memory");
    const int g = lane >> 4, i16 = lane & 15, q = i16 >> 2, p = i16 & 3;
#pragma unroll
    for (int c2 = 0; c2 < 2; ++c2)
#pragma unroll
      for (int s = 0; s < 4; ++s)
#pragma unroll
        for (int nt = 0; nt < 2; ++nt) {
          const int k0 = c2 * 64 + 32 * (g >> 1) + 8 * s;
          char* a1 = sw + (k0 + q) * 128 + (nt * 32 + (g & 1) * 16 + 4 * p) * 2;
          s16x4 lo = __builtin_amdgcn_ds_read_tr16_b64_v4i16((__attribute__((address_space(3))) s16x4*)(a1));
          s16x4 hi = __builtin_amdgcn_ds_read_tr16_b64_v4i16((__attribute__((address_space(3))) s16x4*)(a1 + 512));
          bf16x8 f;
          f[0] = lo[0]; f[1] = lo[1]; f[2] = lo[2]; f[3] = lo[3];
          f[4] = hi[0]; f[5] = hi[1]; f[6] = hi[2]; f[7] = hi[3];
          Bf[half * 2 + c2][s][nt] = f;
        }
  }

  STAMP(2);
  const short* abase;
  if (wv == rank) {
    abase = x + (size_t)(rank * 256) * KB;
  } else {
    abase = (const short*)(myws + par + (size_t)wv * SLOT);
    unsigned* tmo = (unsigned*)(myws + TMO_OFF);
    if (lane == 0) {
      const unsigned long long t0 = __builtin_amdgcn_s_memrealtime();
      for (int pc = 0; pc < 2; ++pc) {
        unsigned* flag = (unsigned*)(myws + FLAG_OFF + (size_t)(wv * NPIECE + rg * 2 + pc) * 128);
        while (__hip_atomic_load(flag, __ATOMIC_RELAXED, __HIP_MEMORY_SCOPE_SYSTEM) < call) {
          if (__hip_atomic_load(tmo, __ATOMIC_RELAXED, __HIP_MEMORY_SCOPE_AGENT) == call ||
              __builtin_amdgcn_s_memrealtime() - t0 > DEADLINE_TICKS) {
            __hip_atomic_store(tmo, call, __ATOMIC_RELAXED, __HIP_MEMORY_SCOPE_AGENT);
            s_bad = 1;
            break;
          }
          __builtin_amdgcn_s_sleep(1);
        }
      }
    }
#define AMODE 1
    __builtin_amdgcn_fence(__ATOMIC_ACQUIRE, "wavefront");
    asm volatile("" ::: "memory");
  }

  STAMP(3);
  const short* ap = abase + (size_t)(rg * 32 + r) * KB + 32 * h;
  bf16x8 Af[4][4];
  const auto arsrc = __builtin_amdgcn_make_buffer_rsrc((void*)abase, 0, (int)SLOT, 0x00020000);
  const int aoff = ((rg * 32 + r) * KB + 32 * h) * 2;
#pragma unroll
  for (int c = 0; c < 4; ++c)
#pragma unroll
    for (int s = 0; s < 4; ++s) {
      {
        u32x4 t = __builtin_amdgcn_raw_buffer_load_b128(arsrc, aoff + (64 * c + 8 * s) * 2, 0, 17);
        Af[c][s] = __builtin_bit_cast(bf16x8, t);
      }
    }

  f32x16 acc0 = {}, acc1 = {};
#pragma unroll
  for (int c = 0; c < 4; ++c)
#pragma unroll
    for (int s = 0; s < 4; ++s) {
      acc0 = __builtin_amdgcn_mfma_f32_32x32x16_bf16(Af[c][s], Bf[c][s][0], acc0, 0, 0, 0);
      acc1 = __builtin_amdgcn_mfma_f32_32x32x16_bf16(Af[c][s], Bf[c][s][1], acc1, 0, 0, 0);
    }

  STAMP(4);
  __syncthreads();
  float* part = (float*)smem;
#pragma unroll
  for (int i = 0; i < 16; ++i) {
    const int row = (i & 3) + 8 * (i >> 2) + 4 * h;
    part[(wv * 32 + row) * 64 + r] = acc0[i];
    part[(wv * 32 + row) * 64 + 32 + r] = acc1[i];
  }
  __syncthreads();
  {
    const int row = tid >> 4, c4 = (tid & 15) * 4;
    f32x4 sum = *(const f32x4*)(part + row * 64 + c4);
#pragma unroll
    for (int k = 1; k < 8; ++k) sum += *(const f32x4*)(part + (k * 32 + row) * 64 + c4);
    const bool bad = s_bad != 0;
    s16x4 o;
#pragma unroll
    for (int e = 0; e < 4; ++e) {
      float y = sum[e];
      float z = fminf(fmaxf(y, -60.f), 60.f);
      float v = y / (1.f + __expf(-z));
      o[e] = bad ? (short)0x7FC0 : (short)f2bf(v);
    }
    *(s16x4*)(out + (size_t)(rg * 32 + row) * NN + n0 + c4) = o;
  }
  STAMP(5);
}

extern "C" void kernel_launch(int rank, int nranks, ncclComm_t comm, hipStream_t stream,
                              void* const* d_in, const int* in_sizes, int n_in, void* d_out,
                              int out_size, const int* in_dtypes, int out_dtype,
                              void* const* d_out_all, void* const* d_ws_all, size_t ws_bytes) {
  (void)comm; (void)in_sizes; (void)n_in; (void)out_size; (void)in_dtypes; (void)out_dtype;
  (void)d_out_all; (void)ws_bytes; (void)nranks;
  hipLaunchKernelGGL(a2a_gemm_kernel, dim3(256), dim3(512), 0, stream, rank,
                     (const short*)d_in[0], (const short*)d_in[1], (short*)d_out, d_ws_all);
}
```
